```python
import numpy as np

kernel_name = "gpu_a2a_gemm_m4096_k4096_n16384_bf16_qe4m3"


ABSMAX_RTOL = 0.09

COMPUTE_DTYPE = "bfloat16"
EPILOGUE = "quant_e4m3"
IN_QUANT = None
OUT_QUANT = "e4m3"
IN_DTYPES = {"x": "bfloat16", "w_mat": "bfloat16"}
WORLD_SIZE = 8

def _to_bf16(x):
    u = np.ascontiguousarray(x, dtype=np.float32).view(np.uint32)
    return ((u + 0x7FFF + ((u >> 16) & 1)) & 0xFFFF0000).view(np.float32)

def setup_inputs():

    x = _to_bf16(np.random.randn(4096, 4096).astype(np.float32))
    w_mat = _to_bf16((np.random.randn(4096, 16384) / np.sqrt(4096)).astype(np.float32))
    return {"x": x, "w_mat": w_mat}

def _snap_fp8(x, mant_bits, max_abs, min_normal):
    a = np.abs(x.astype(np.float64))
    _m, _e = np.frexp(a)
    q = np.where(a >= min_normal, np.ldexp(1.0, _e - 1 - mant_bits),
                 min_normal * 2.0 ** -mant_bits)
    snapped = np.minimum(np.round(a / q) * q, max_abs)
    return (np.sign(x) * snapped).astype(np.float32)
def _quant_dequant(y, amax):
    scale = amax / 448.0
    q = _snap_fp8(y / scale, 3, 448.0, 2.0 ** -6)
    return (q * scale).astype(np.float32)
def _epilogue(y):
    return _quant_dequant(y, np.abs(y).max())

def reference(x, w_mat):
    y = (x @ w_mat).astype(np.float32)
    return _epilogue(y)
```

```cpp
#include <hip/hip_runtime.h>
#include <rccl/rccl.h>
#include <cstdio>
#include <cstdint>

#define HIP_CHECK(expr) do { hipError_t _e = (expr); if (_e != hipSuccess) { \
    fprintf(stderr, "HIP error %s at %s:%d\n", hipGetErrorString(_e), __FILE__, __LINE__); \
    return; } } while (0)

namespace {

typedef short bf16x8 __attribute__((ext_vector_type(8)));
typedef short bf16x4 __attribute__((ext_vector_type(4)));
typedef float f32x4 __attribute__((ext_vector_type(4)));
typedef unsigned u32x4 __attribute__((ext_vector_type(4)));
#define LAS __attribute__((address_space(3)))

constexpr int NR = 8;
constexpr int MLOC = 512;
constexpr int KLOC = 512;
constexpr int KTOT = 4096;
constexpr int NTOT = 16384;
constexpr int BM = 256, BN = 128, BK = 64;
constexpr int NWG = (MLOC / BM) * (NTOT / BN);
constexpr int NTHR = 512;
constexpr int NKT = KTOT / BK;
constexpr int KT_PER_CHUNK = KLOC / BK;
constexpr int A_BYTES = BM * BK * 2;
constexpr int B_BYTES = BK * BN * 2;
constexpr int STAGE_BYTES = A_BYTES + B_BYTES;
constexpr int NSTAGE = 3;
constexpr int LDS_BYTES = NSTAGE * STAGE_BYTES + 256;

constexpr size_t WS_TICKET = 0;
constexpr size_t WS_TMO = 128;
constexpr size_t WS_AMAX = 256;
constexpr size_t WS_DONE = 384;
constexpr size_t WS_XFLAG = 1024;
constexpr size_t WS_ASLOT = 4096;
constexpr size_t WS_XSTAGE = 65536;
constexpr size_t WS_NEEDED = WS_XSTAGE + (size_t)NR * MLOC * KLOC * 2;

constexpr unsigned long long DEADLINE_TICKS = 100000000ull;

__device__ __forceinline__ int a_lds_byte(int r, int c) {
    int st = (r >> 4) * 2 + (c >> 5), rr = r & 15, cc = c & 31, ob = rr * 64 + cc * 2;
    return st * 1024 + (ob ^ (((ob >> 9) & 1) << 5));
}
__device__ __forceinline__ void a_stage_rc(int b, int& R, int& C) {
    int st = b >> 10, sb = b & 1023, swz = sb ^ (((sb >> 9) & 1) << 5);
    R = (st >> 1) * 16 + (swz >> 6);
    C = (st & 1) * 32 + ((swz & 63) >> 1);
}

__device__ __forceinline__ unsigned ld_relaxed_sys(const unsigned* p) {
    return __hip_atomic_load(p, __ATOMIC_RELAXED, __HIP_MEMORY_SCOPE_SYSTEM);
}

struct Params {
    const unsigned short* x;
    const unsigned short* w;
    unsigned char* out;
    char* ws_all[NR];
    int rank;
};

__global__ __launch_bounds__(NTHR, 2) void fused_kernel(Params p) {
    extern __shared__ __attribute__((aligned(16))) char smem[];
    LAS unsigned* ctl = (LAS unsigned*)(smem + NSTAGE * STAGE_BYTES);
    const int tid = threadIdx.x;
    const int bid = blockIdx.x;
    const int rank = p.rank;
    char* ws = p.ws_all[rank];
    unsigned* tmo = (unsigned*)(ws + WS_TMO);

    if (tid == 0) {
        unsigned t = __hip_atomic_fetch_add((unsigned*)(ws + WS_TICKET), 1u, __ATOMIC_RELAXED,
                                            __HIP_MEMORY_SCOPE_AGENT);
        ctl[0] = t / NWG + 1;
    }
    __syncthreads();
    const unsigned call = ctl[0];

    {
        const int grp = bid >> 5;
        if (grp != rank) {
            const int r0 = grp * MLOC + (bid & 31) * 16;
            const u32x4* src = (const u32x4*)(p.x + (size_t)r0 * KLOC);
            u32x4* dst = (u32x4*)(p.ws_all[grp] + WS_XSTAGE +
                                  ((size_t)rank * MLOC + (bid & 31) * 16) * (KLOC * 2));
            u32x4 v0 = src[tid], v1 = src[tid + NTHR];
            dst[tid] = v0;
            dst[tid + NTHR] = v1;
            asm volatile("s_waitcnt vmcnt(0)" ::: "memory");
            __syncthreads();
            if (tid == 0) {
                __builtin_amdgcn_fence(__ATOMIC_RELEASE, "");
                asm volatile("s_waitcnt vmcnt(0)" ::: "memory");
                __hip_atomic_fetch_add((unsigned*)(p.ws_all[grp] + WS_XFLAG + 128 * rank), 1u,
                                       __ATOMIC_RELAXED, __HIP_MEMORY_SCOPE_SYSTEM);
                asm volatile("s_waitcnt vmcnt(0)" ::: "memory");
            }
        }
    }

    const int xcd = bid & 7, idx = bid >> 3;
    const int pn = xcd * 16 + (idx >> 1), pm = idx & 1;
    const int brow = pm * BM, bcol = pn * BN;
    const int wid = tid >> 6, lane = tid & 63;
    const int wr = wid >> 1, wc = wid & 1;
    const int fr = lane & 15, fq = lane >> 4;

    int a_src[4];
#pragma unroll
    for (int i = 0; i < 4; ++i) {
        int R, C;
        a_stage_rc(tid * 16 + i * 8192, R, C);
        a_src[i] = (brow + R) * KLOC + C;
    }
    int b_src[2];
#pragma unroll
    for (int i = 0; i < 2; ++i) {
        int b = tid * 16 + i * 8192;
        int sub = b >> 8, rowp = (b >> 5) & 7, half = (b >> 4) & 1;
        int kb = sub >> 3, nb = sub & 7;
        int ks = rowp ^ ((kb & 1) << 2);
        b_src[i] = (kb * 8 + ks) * NTOT + bcol + nb * 16 + half * 8;
    }

    auto stage = [&](int t) {
        const int c = t >> 3, src = (rank + c) & (NR - 1);
        const unsigned short* abase =
            (c == 0) ? (p.x + (size_t)rank * MLOC * KLOC)
                     : (const unsigned short*)(ws + WS_XSTAGE) + (size_t)src * MLOC * KLOC;
        abase += (t & 7) * BK;
        const unsigned short* bbase = p.w + (size_t)(src * KLOC + (t & 7) * BK) * NTOT;
        char* sa = smem + (t % NSTAGE) * STAGE_BYTES;
        char* sb = sa + A_BYTES;
#pragma unroll
        for (int i = 0; i < 4; ++i)
            __builtin_amdgcn_global_load_lds((const unsigned*)(abase + a_src[i]),
                                             (unsigned*)(sa + tid * 16 + i * 8192), 16, 0, 0);
#pragma unroll
        for (int i = 0; i < 2; ++i)
            __builtin_amdgcn_global_load_lds((const unsigned*)(bbase + b_src[i]),
                                             (unsigned*)(sb + tid * 16 + i * 8192), 16, 0, 0);
    };

    auto wait_chunk = [&](int c) {
        const int src = (rank + c) & (NR - 1);
        const unsigned* flag = (const unsigned*)(ws + WS_XFLAG + 128 * src);
        const unsigned need = 32u * call;
        unsigned long long t0 = __builtin_amdgcn_s_memrealtime();
        while (ld_relaxed_sys(flag) < need) {
            if (__hip_atomic_load(tmo, __ATOMIC_RELAXED, __HIP_MEMORY_SCOPE_AGENT) == call ||
                __builtin_amdgcn_s_memrealtime() - t0 > DEADLINE_TICKS) {
                __hip_atomic_store(tmo, call, __ATOMIC_RELAXED, __HIP_MEMORY_SCOPE_AGENT);
                break;
            }
            __builtin_amdgcn_s_sleep(2);
        }
        __builtin_amdgcn_fence(__ATOMIC_ACQUIRE, "");
    };

    f32x4 acc[4][4];
#pragma unroll
    for (int m = 0; m < 4; ++m)
#pragma unroll
        for (int n = 0; n < 4; ++n) acc[m][n] = (f32x4){0.f, 0.f, 0.f, 0.f};

    stage(0);
    stage(1);
    for (int t = 0; t < NKT; ++t) {
        if (t + 1 < NKT) asm volatile("s_waitcnt vmcnt(6)" ::: "memory");
        else asm volatile("s_waitcnt vmcnt(0)" ::: "memory");
        if (((t + 2) & 7) == 0 && t + 2 < NKT) {
            if (tid == 0) wait_chunk((t + 2) >> 3);
            asm volatile("s_waitcnt vmcnt(0)" ::: "memory");
        }
        __builtin_amdgcn_s_barrier();
        if (t + 2 < NKT) stage(t + 2);
        const char* sa = smem + (t % NSTAGE) * STAGE_BYTES;
        const char* sb = sa + A_BYTES;
#pragma unroll
        for (int ks = 0; ks < 2; ++ks) {
            bf16x8 af[4], bfr[4];
#pragma unroll
            for (int m = 0; m < 4; ++m)
                af[m] = *(const bf16x8*)(sa + a_lds_byte(wr * 64 + m * 16 + fr, ks * 32 + fq * 8));
#pragma unroll
            for (int n = 0; n < 4; ++n) {
                const int kb = ks * 4 + fq, nb = wc * 4 + n;
                const int q = (lane & 15) >> 2, pp = lane & 3;
                const int sw = (kb & 1) << 2;
                const char* base = sb + (kb * 8 + nb) * 256 + pp * 8;
                bf16x4 lo = __builtin_amdgcn_ds_read_tr16_b64_v4i16((LAS bf16x4*)(base + ((q) ^ sw) * 32));
                bf16x4 hi = __builtin_amdgcn_ds_read_tr16_b64_v4i16((LAS bf16x4*)(base + ((4 + q) ^ sw) * 32));
                bfr[n] = (bf16x8){lo[0], lo[1], lo[2], lo[3], hi[0], hi[1], hi[2], hi[3]};
            }
#pragma unroll
            for (int m = 0; m < 4; ++m)
#pragma unroll
                for (int n = 0; n < 4; ++n)
                    acc[m][n] = __builtin_amdgcn_mfma_f32_16x16x32_bf16(bfr[n], af[m], acc[m][n], 0, 0, 0);
        }
    }

    float mx = 0.f;
#pragma unroll
    for (int m = 0; m < 4; ++m)
#pragma unroll
        for (int n = 0; n < 4; ++n)
#pragma unroll
            for (int j = 0; j < 4; ++j) mx = fmaxf(mx, fabsf(acc[m][n][j]));
#pragma unroll
    for (int o = 32; o > 0; o >>= 1) mx = fmaxf(mx, __shfl_xor(mx, o));
    __syncthreads();
    if (lane == 0) ctl[8 + wid] = __float_as_uint(mx);
    __syncthreads();
    if (tid == 0) {
        unsigned m = ctl[8];
        for (int i = 1; i < 8; ++i) m = max(m, ctl[8 + i]);
        unsigned long long v = ((unsigned long long)call << 32) | m;
        __hip_atomic_fetch_max((unsigned long long*)(ws + WS_AMAX), v, __ATOMIC_RELAXED,
                               __HIP_MEMORY_SCOPE_AGENT);
        unsigned tk = __hip_atomic_fetch_add((unsigned*)(ws + WS_DONE), 1u, __ATOMIC_ACQ_REL,
                                             __HIP_MEMORY_SCOPE_AGENT);
        if (tk % NWG == NWG - 1) {
            unsigned long long g = __hip_atomic_load((unsigned long long*)(ws + WS_AMAX),
                                                     __ATOMIC_RELAXED, __HIP_MEMORY_SCOPE_AGENT);
            for (int r = 0; r < NR; ++r)
                __hip_atomic_store((unsigned long long*)(p.ws_all[r] + WS_ASLOT + 128 * rank), g,
                                   __ATOMIC_RELAXED, __HIP_MEMORY_SCOPE_SYSTEM);
        }
        unsigned gm = 0;
        unsigned long long t0 = __builtin_amdgcn_s_memrealtime();
        bool bad = false;
        for (int r = 0; r < NR; ++r) {
            const unsigned long long* slot = (const unsigned long long*)(ws + WS_ASLOT + 128 * r);
            unsigned long long v2;
            while (((v2 = __hip_atomic_load(slot, __ATOMIC_RELAXED, __HIP_MEMORY_SCOPE_SYSTEM)) >> 32) < call) {
                if (__hip_atomic_load(tmo, __ATOMIC_RELAXED, __HIP_MEMORY_SCOPE_AGENT) == call ||
                    __builtin_amdgcn_s_memrealtime() - t0 > DEADLINE_TICKS) {
                    __hip_atomic_store(tmo, call, __ATOMIC_RELAXED, __HIP_MEMORY_SCOPE_AGENT);
                    bad = true;
                    break;
                }
                __builtin_amdgcn_s_sleep(1);
            }
            gm = max(gm, (unsigned)v2);
        }
        if (__hip_atomic_load(tmo, __ATOMIC_RELAXED, __HIP_MEMORY_SCOPE_AGENT) == call) bad = true;
        ctl[1] = gm;
        ctl[2] = bad ? 1u : 0u;
    }
    __syncthreads();
    const float amax = __uint_as_float(ctl[1]);
    const bool bad = ctl[2] != 0;
    const float scale = amax / 448.0f;

#pragma unroll
    for (int m = 0; m < 4; ++m) {
        unsigned char* rowp = p.out + (size_t)(brow + wr * 64 + m * 16 + fr) * NTOT + bcol + wc * 64 + fq * 4;
#pragma unroll
        for (int n = 0; n < 4; ++n) {
            float q0 = acc[m][n][0] / scale, q1 = acc[m][n][1] / scale;
            float q2 = acc[m][n][2] / scale, q3 = acc[m][n][3] / scale;
            int pk = __builtin_amdgcn_cvt_pk_fp8_f32(q0, q1, 0, false);
            pk = __builtin_amdgcn_cvt_pk_fp8_f32(q2, q3, pk, true);
            if (bad) pk = 0x7f7f7f7f;
            *(int*)(rowp + n * 16) = pk;
        }
    }
}

}

extern "C" void kernel_launch(int rank, int nranks, ncclComm_t comm, hipStream_t stream,
                              void* const* d_in, const int* in_sizes, int n_in, void* d_out,
                              int out_size, const int* in_dtypes, int out_dtype,
                              void* const* d_out_all, void* const* d_ws_all, size_t ws_bytes) {
    (void)comm; (void)in_sizes; (void)n_in; (void)out_size; (void)in_dtypes; (void)out_dtype;
    (void)d_out_all;
    if (nranks != NR || ws_bytes < WS_NEEDED) {
        fprintf(stderr, "unexpected configuration: nranks %d ws_bytes %zu\n", nranks, ws_bytes);
        return;
    }
    static bool attr_set = false;
    if (!attr_set) {
        HIP_CHECK(hipFuncSetAttribute((const void*)fused_kernel,
                                      hipFuncAttributeMaxDynamicSharedMemorySize, LDS_BYTES));
        attr_set = true;
    }
    Params p;
    p.x = (const unsigned short*)d_in[0];
    p.w = (const unsigned short*)d_in[1];
    p.out = (unsigned char*)d_out;
    for (int r = 0; r < NR; ++r) p.ws_all[r] = (char*)d_ws_all[r];
    p.rank = rank;
    fused_kernel<<<NWG, NTHR, LDS_BYTES, stream>>>(p);
    HIP_CHECK(hipGetLastError());
}
```

```cpp
#include <hip/hip_runtime.h>
#include <rccl/rccl.h>
#include <cstdio>
#include <cstdint>

#define HIP_CHECK(expr) do { hipError_t _e = (expr); if (_e != hipSuccess) { \
    fprintf(stderr, "HIP error %s at %s:%d\n", hipGetErrorString(_e), __FILE__, __LINE__); \
    return; } } while (0)

namespace {

typedef short bf16x8 __attribute__((ext_vector_type(8)));
typedef short bf16x4 __attribute__((ext_vector_type(4)));
typedef float f32x4 __attribute__((ext_vector_type(4)));
typedef unsigned u32x4 __attribute__((ext_vector_type(4)));
#define LAS __attribute__((address_space(3)))

constexpr int NR = 8;
constexpr int MLOC = 512;
constexpr int KLOC = 512;
constexpr int KTOT = 4096;
constexpr int NTOT = 16384;
constexpr int BM = 256, BN = 128, BK = 64;
constexpr int NWG = (MLOC / BM) * (NTOT / BN);
constexpr int NTHR = 512;
constexpr int NKT = KTOT / BK;
constexpr int KT_PER_CHUNK = KLOC / BK;
constexpr int A_BYTES = BM * BK * 2;
constexpr int B_BYTES = BK * BN * 2;
constexpr int STAGE_BYTES = A_BYTES + B_BYTES;
constexpr int NSTAGE = 3;
constexpr int LDS_BYTES = NSTAGE * STAGE_BYTES + 256;

constexpr size_t WS_TICKET = 0;
constexpr size_t WS_TMO = 128;
constexpr size_t WS_AMAX = 256;
constexpr size_t WS_DONE = 384;
constexpr size_t WS_XFLAG = 1024;
constexpr size_t WS_ASLOT = 4096;
constexpr size_t WS_XSTAGE = 65536;
constexpr size_t WS_NEEDED = WS_XSTAGE + (size_t)NR * MLOC * KLOC * 2;

constexpr unsigned long long DEADLINE_TICKS = 100000000ull;

__device__ __forceinline__ int a_lds_byte(int r, int c) { return r * 128 + (((c >> 3) ^ (r & 7)) << 4); }
__device__ __forceinline__ void a_stage_rc(int b, int& R, int& C) {
    R = b >> 7;
    C = (((b >> 4) & 7) ^ (R & 7)) << 3;
}

__device__ __forceinline__ unsigned ld_relaxed_sys(const unsigned* p) {
    return __hip_atomic_load(p, __ATOMIC_RELAXED, __HIP_MEMORY_SCOPE_SYSTEM);
}

struct Params {
    const unsigned short* x;
    const unsigned short* w;
    unsigned char* out;
    char* ws_all[NR];
    int rank;
};

__global__ __launch_bounds__(NTHR, 2) void fused_kernel(Params p) {
    extern __shared__ __attribute__((aligned(16))) char smem[];
    LAS unsigned* ctl = (LAS unsigned*)(smem + NSTAGE * STAGE_BYTES);
    const int tid = threadIdx.x;
    const int bid = blockIdx.x;
    const int rank = p.rank;
    char* ws = p.ws_all[rank];
    unsigned* tmo = (unsigned*)(ws + WS_TMO);

    if (tid == 0) {
        unsigned t = __hip_atomic_fetch_add((unsigned*)(ws + WS_TICKET), 1u, __ATOMIC_RELAXED,
                                            __HIP_MEMORY_SCOPE_AGENT);
        ctl[0] = t / NWG + 1;
    }
    __syncthreads();
    const unsigned call = ctl[0];

    {
        const int grp = bid >> 5;
        if (grp != rank) {
            const int r0 = grp * MLOC + (bid & 31) * 16;
            const u32x4* src = (const u32x4*)(p.x + (size_t)r0 * KLOC);
            u32x4* dst = (u32x4*)(p.ws_all[grp] + WS_XSTAGE +
                                  ((size_t)rank * MLOC + (bid & 31) * 16) * (KLOC * 2));
            u32x4 v0 = src[tid], v1 = src[tid + NTHR];
            dst[tid] = v0;
            dst[tid + NTHR] = v1;
            asm volatile("s_waitcnt vmcnt(0)" ::: "memory");
            __syncthreads();
            if (tid == 0) {
                __builtin_amdgcn_fence(__ATOMIC_RELEASE, "");
                asm volatile("s_waitcnt vmcnt(0)" ::: "memory");
                __hip_atomic_fetch_add((unsigned*)(p.ws_all[grp] + WS_XFLAG + 128 * rank), 1u,
                                       __ATOMIC_RELAXED, __HIP_MEMORY_SCOPE_SYSTEM);
                asm volatile("s_waitcnt vmcnt(0)" ::: "memory");
            }
        }
    }

    const int xcd = bid & 7, idx = bid >> 3;
    const int pn = xcd * 16 + (idx >> 1), pm = idx & 1;
    const int brow = pm * BM, bcol = pn * BN;
    const int wid = tid >> 6, lane = tid & 63;
    const int wr = wid >> 1, wc = wid & 1;
    const int fr = lane & 15, fq = lane >> 4;

    int a_src[4];
#pragma unroll
    for (int i = 0; i < 4; ++i) {
        int R, C;
        a_stage_rc(tid * 16 + i * 8192, R, C);
        a_src[i] = (brow + R) * KLOC + C;
    }
    int b_src[2];
#pragma unroll
    for (int i = 0; i < 2; ++i) {
        int b = tid * 16 + i * 8192;
        int sub = b >> 8, rowp = (b >> 5) & 7, half = (b >> 4) & 1;
        int kb = sub >> 3, nb = sub & 7;
        int ks = rowp ^ ((kb & 1) << 2);
        b_src[i] = (kb * 8 + ks) * NTOT + bcol + nb * 16 + half * 8;
    }

    auto stage = [&](int t) {
        const int c = t >> 3, src = (rank + c) & (NR - 1);
        const unsigned short* abase =
            (c == 0) ? (p.x + (size_t)rank * MLOC * KLOC)
                     : (const unsigned short*)(ws + WS_XSTAGE) + (size_t)src * MLOC * KLOC;
        abase += (t & 7) * BK;
        const unsigned short* bbase = p.w + (size_t)(src * KLOC + (t & 7) * BK) * NTOT;
        char* sa = smem + (t % NSTAGE) * STAGE_BYTES;
        char* sb = sa + A_BYTES;
#pragma unroll
        for (int i = 0; i < 4; ++i)
            __builtin_amdgcn_global_load_lds((const unsigned*)(abase + a_src[i]),
                                             (unsigned*)(sa + tid * 16 + i * 8192), 16, 0, 0);
#pragma unroll
        for (int i = 0; i < 2; ++i)
            __builtin_amdgcn_global_load_lds((const unsigned*)(bbase + b_src[i]),
                                             (unsigned*)(sb + tid * 16 + i * 8192), 16, 0, 0);
    };

    auto wait_chunk = [&](int c) {
        const int src = (rank + c) & (NR - 1);
        const unsigned* flag = (const unsigned*)(ws + WS_XFLAG + 128 * src);
        const unsigned need = 32u * call;
        unsigned long long t0 = __builtin_amdgcn_s_memrealtime();
        while (ld_relaxed_sys(flag) < need) {
            if (__hip_atomic_load(tmo, __ATOMIC_RELAXED, __HIP_MEMORY_SCOPE_AGENT) == call ||
                __builtin_amdgcn_s_memrealtime() - t0 > DEADLINE_TICKS) {
                __hip_atomic_store(tmo, call, __ATOMIC_RELAXED, __HIP_MEMORY_SCOPE_AGENT);
                break;
            }
            __builtin_amdgcn_s_sleep(2);
        }
        __builtin_amdgcn_fence(__ATOMIC_ACQUIRE, "");
    };

    const unsigned lds_base = (unsigned)(size_t)(LAS char*)smem;
    const unsigned a_rdk[2] = {(unsigned)a_lds_byte(wr * 64 + fr, fq * 8), (unsigned)a_lds_byte(wr * 64 + fr, 32 + fq * 8)};
    const unsigned b_sw = (unsigned)(fq & 1) << 2;
    const unsigned b_rd0 = (unsigned)((fq * 8 + wc * 4) * 256 + (lane & 3) * 8) + ((((lane & 15) >> 2)) ^ b_sw) * 32;
    const unsigned b_rd1 = (unsigned)((fq * 8 + wc * 4) * 256 + (lane & 3) * 8) + ((4 + ((lane & 15) >> 2)) ^ b_sw) * 32;
    f32x4 acc[4][4];
#pragma unroll
    for (int m = 0; m < 4; ++m)
#pragma unroll
        for (int n = 0; n < 4; ++n) acc[m][n] = (f32x4){0.f, 0.f, 0.f, 0.f};

    stage(0);
    stage(1);
    for (int t = 0; t < NKT; ++t) {
        if (t + 1 < NKT) asm volatile("s_waitcnt vmcnt(6)" ::: "memory");
        else asm volatile("s_waitcnt vmcnt(0)" ::: "memory");
        if (((t + 2) & 7) == 0 && t + 2 < NKT) {
            if (tid == 0) wait_chunk((t + 2) >> 3);
            asm volatile("s_waitcnt vmcnt(0)" ::: "memory");
        }
        __builtin_amdgcn_s_barrier();
        if (t + 2 < NKT) stage(t + 2);
        const unsigned sa = lds_base + (t % NSTAGE) * STAGE_BYTES;
        const unsigned sb = sa + A_BYTES;
        bf16x8 af[2][4];
        bf16x4 bl[2][4], bh[2][4];
#pragma unroll
        for (int ks = 0; ks < 2; ++ks) {
#pragma unroll
            for (int n = 0; n < 4; ++n) {
                asm volatile("ds_read_b64_tr_b16 %0, %1 offset:%2" : "=v"(bl[ks][n]) : "v"(sb + b_rd0), "i"(ks * 8192 + n * 256));
                asm volatile("ds_read_b64_tr_b16 %0, %1 offset:%2" : "=v"(bh[ks][n]) : "v"(sb + b_rd1), "i"(ks * 8192 + n * 256));
            }
#pragma unroll
            for (int m = 0; m < 4; ++m)
                asm volatile("ds_read_b128 %0, %1 offset:%2" : "=v"(af[ks][m]) : "v"(sa + a_rdk[ks]), "i"(m * 2048));
        }
#pragma unroll
        for (int ks = 0; ks < 2; ++ks) {
            if (ks == 0) asm volatile("s_waitcnt lgkmcnt(12)" ::: "memory");
            else asm volatile("s_waitcnt lgkmcnt(0)" ::: "memory");
            __builtin_amdgcn_sched_barrier(0);
            __builtin_amdgcn_s_setprio(1);
#pragma unroll
            for (int m = 0; m < 4; ++m)
#pragma unroll
                for (int n = 0; n < 4; ++n) {
                    bf16x8 bfr = (bf16x8){bl[ks][n][0], bl[ks][n][1], bl[ks][n][2], bl[ks][n][3],
                                          bh[ks][n][0], bh[ks][n][1], bh[ks][n][2], bh[ks][n][3]};
                    acc[m][n] = __builtin_amdgcn_mfma_f32_16x16x32_bf16(bfr, af[ks][m], acc[m][n], 0, 0, 0);
                }
            __builtin_amdgcn_s_setprio(0);
            __builtin_amdgcn_sched_barrier(0);
        }
    }

    float mx = 0.f;
#pragma unroll
    for (int m = 0; m < 4; ++m)
#pragma unroll
        for (int n = 0; n < 4; ++n)
#pragma unroll
            for (int j = 0; j < 4; ++j) mx = fmaxf(mx, fabsf(acc[m][n][j]));
#pragma unroll
    for (int o = 32; o > 0; o >>= 1) mx = fmaxf(mx, __shfl_xor(mx, o));
    __syncthreads();
    if (lane == 0) ctl[8 + wid] = __float_as_uint(mx);
    __syncthreads();
    if (tid == 0) {
        unsigned m = ctl[8];
        for (int i = 1; i < 8; ++i) m = max(m, ctl[8 + i]);
        unsigned long long v = ((unsigned long long)call << 32) | m;
        __hip_atomic_fetch_max((unsigned long long*)(ws + WS_AMAX), v, __ATOMIC_RELAXED,
                               __HIP_MEMORY_SCOPE_AGENT);
        unsigned tk = __hip_atomic_fetch_add((unsigned*)(ws + WS_DONE), 1u, __ATOMIC_ACQ_REL,
                                             __HIP_MEMORY_SCOPE_AGENT);
        if (tk % NWG == NWG - 1) {
            unsigned long long g = __hip_atomic_load((unsigned long long*)(ws + WS_AMAX),
                                                     __ATOMIC_RELAXED, __HIP_MEMORY_SCOPE_AGENT);
            for (int r = 0; r < NR; ++r)
                __hip_atomic_store((unsigned long long*)(p.ws_all[r] + WS_ASLOT + 128 * rank), g,
                                   __ATOMIC_RELAXED, __HIP_MEMORY_SCOPE_SYSTEM);
        }
        unsigned gm = 0;
        unsigned long long t0 = __builtin_amdgcn_s_memrealtime();
        bool bad = false;
        for (int r = 0; r < NR; ++r) {
            const unsigned long long* slot = (const unsigned long long*)(ws + WS_ASLOT + 128 * r);
            unsigned long long v2;
            while (((v2 = __hip_atomic_load(slot, __ATOMIC_RELAXED, __HIP_MEMORY_SCOPE_SYSTEM)) >> 32) < call) {
                if (__hip_atomic_load(tmo, __ATOMIC_RELAXED, __HIP_MEMORY_SCOPE_AGENT) == call ||
                    __builtin_amdgcn_s_memrealtime() - t0 > DEADLINE_TICKS) {
                    __hip_atomic_store(tmo, call, __ATOMIC_RELAXED, __HIP_MEMORY_SCOPE_AGENT);
                    bad = true;
                    break;
                }
                __builtin_amdgcn_s_sleep(1);
            }
            gm = max(gm, (unsigned)v2);
        }
        if (__hip_atomic_load(tmo, __ATOMIC_RELAXED, __HIP_MEMORY_SCOPE_AGENT) == call) bad = true;
        ctl[1] = gm;
        ctl[2] = bad ? 1u : 0u;
    }
    __syncthreads();
    const float amax = __uint_as_float(ctl[1]);
    const bool bad = ctl[2] != 0;
    const float scale = amax / 448.0f;

#pragma unroll
    for (int m = 0; m < 4; ++m) {
        unsigned char* rowp = p.out + (size_t)(brow + wr * 64 + m * 16 + fr) * NTOT + bcol + wc * 64 + fq * 4;
#pragma unroll
        for (int n = 0; n < 4; ++n) {
            float q0 = acc[m][n][0] / scale, q1 = acc[m][n][1] / scale;
            float q2 = acc[m][n][2] / scale, q3 = acc[m][n][3] / scale;
            int pk = __builtin_amdgcn_cvt_pk_fp8_f32(q0, q1, 0, false);
            pk = __builtin_amdgcn_cvt_pk_fp8_f32(q2, q3, pk, true);
            if (bad) pk = 0x7f7f7f7f;
            *(int*)(rowp + n * 16) = pk;
        }
    }
}

}

extern "C" void kernel_launch(int rank, int nranks, ncclComm_t comm, hipStream_t stream,
                              void* const* d_in, const int* in_sizes, int n_in, void* d_out,
                              int out_size, const int* in_dtypes, int out_dtype,
                              void* const* d_out_all, void* const* d_ws_all, size_t ws_bytes) {
    (void)comm; (void)in_sizes; (void)n_in; (void)out_size; (void)in_dtypes; (void)out_dtype;
    (void)d_out_all;
    if (nranks != NR || ws_bytes < WS_NEEDED) {
        fprintf(stderr, "unexpected configuration: nranks %d ws_bytes %zu\n", nranks, ws_bytes);
        return;
    }
    static bool attr_set = false;
    if (!attr_set) {
        HIP_CHECK(hipFuncSetAttribute((const void*)fused_kernel,
                                      hipFuncAttributeMaxDynamicSharedMemorySize, LDS_BYTES));
        attr_set = true;
    }
    Params p;
    p.x = (const unsigned short*)d_in[0];
    p.w = (const unsigned short*)d_in[1];
    p.out = (unsigned char*)d_out;
    for (int r = 0; r < NR; ++r) p.ws_all[r] = (char*)d_ws_all[r];
    p.rank = rank;
    fused_kernel<<<NWG, NTHR, LDS_BYTES, stream>>>(p);
    HIP_CHECK(hipGetLastError());
}
```
